```python
import numpy as np

kernel_name = "gpu_gemm_a2a_m4096_k4096_n16384_e4m3e4m3_dqrelu"


ABSMAX_RTOL = 0.03

COMPUTE_DTYPE = "float32"
EPILOGUE = "dequant_relu"
IN_QUANT = None
OUT_QUANT = None
IN_FP8 = "e4m3e4m3"
IN_DTYPES = {"x": "e4m3", "w_mat": "e4m3",
             "scale_x": "float32", "scale_w": "float32"}
WORLD_SIZE = 8

def _snap_fp8(x, mant_bits, max_abs, min_normal):
    a = np.abs(x.astype(np.float64))
    _m, _e = np.frexp(a)
    q = np.where(a >= min_normal, np.ldexp(1.0, _e - 1 - mant_bits),
                 min_normal * 2.0 ** -mant_bits)
    snapped = np.minimum(np.round(a / q) * q, max_abs)
    return (np.sign(x) * snapped).astype(np.float32)

def setup_inputs():

    x = _snap_fp8(np.random.randn(4096, 4096).astype(np.float32), 3, 448.0, 2.0 ** -6)
    w_mat = _snap_fp8(np.random.randn(4096, 16384).astype(np.float32), 3, 448.0, 2.0 ** -6)
    scale_x = (0.02 * (1.0 + np.random.rand(1))).astype(np.float32)
    scale_w = (0.02 * (1.0 + np.random.rand(1))).astype(np.float32)
    return {"x": x, "w_mat": w_mat, "scale_x": scale_x, "scale_w": scale_w}

def reference(x, w_mat, scale_x, scale_w):
    acc = (x @ w_mat).astype(np.float32)
    y = (acc * (scale_x * scale_w)).astype(np.float32)
    return np.maximum(y, 0.0).astype(np.float32)
```

```cpp
#include <hip/hip_runtime.h>
#include <rccl/rccl.h>
#include <cstdio>
#include <type_traits>

#define HIP_CHECK(expr) do { hipError_t _e = (expr); if (_e != hipSuccess) { \
    fprintf(stderr, "HIP error %s at %s:%d\n", hipGetErrorString(_e), __FILE__, __LINE__); \
    return; } } while (0)

namespace {

typedef int v8i __attribute__((ext_vector_type(8)));
typedef int v4i __attribute__((ext_vector_type(4)));
typedef int v2i __attribute__((ext_vector_type(2)));
typedef float v16f __attribute__((ext_vector_type(16)));
typedef unsigned int u32;
typedef unsigned char u8;

constexpr int NR = 8;
constexpr int KD = 4096;
constexpr int ND = 16384;
constexpr int MS = 512;
constexpr int NS = 2048;
constexpr int BM = 256, BX = 32, BN = 128, BK = 128;
constexpr int NT = KD / BK;
constexpr int NTHREADS = 512;
constexpr int NWG = 256;
constexpr int KCH = 256;
constexpr int NCH = KD / KCH;
constexpr int SPC = KCH / BK;
constexpr int STAGE_A = BM * BK;
constexpr int STAGE_B = BK * BN;
constexpr int STAGE_X = BX * BK;
constexpr int STAGE = STAGE_A + STAGE_B;
constexpr int STAGE1 = STAGE_B + STAGE_X;
constexpr int NSTAGE1 = 7;
constexpr int NSTAGE = 3;
constexpr int LDS_CTL = NSTAGE * STAGE;
constexpr int LDS_MIR = LDS_CTL + 64;
constexpr int LDS_BYTES = LDS_MIR + 1024;

constexpr size_t WS_CNT = 0;
constexpr size_t WS_TMO = 4096;
constexpr size_t WS_FLAG = 8192;
constexpr size_t WS_STAGE = 1ull << 20;
constexpr size_t SHARD_BYTES = (size_t)MS * KD;
constexpr size_t SLOT_BYTES = (size_t)NR * SHARD_BYTES;
constexpr size_t WS_NEED = WS_STAGE + 2 * SLOT_BYTES;

constexpr unsigned long long DEADLINE_TICKS = 40000000ull;

struct Params {
    const u8* x;
    const u8* w;
    const float* sx;
    const float* sw;
    float* out;
    u8* ws[NR];
    int rank;
};

__device__ __forceinline__ void st16_sys(u8* p, v4i v) {
    asm volatile("global_store_dwordx4 %0, %1, off sc0 sc1\n\ts_nop 1" :: "v"(p), "v"(v) : "memory");
}

#define STAMP(i) do { } while (0)
#define WAITV(n) asm volatile("s_waitcnt vmcnt(" #n ")" ::: "memory")

__device__ __forceinline__ void glds16(const u8* g, u8* l) {
    __builtin_amdgcn_global_load_lds((const u32*)g, (u32*)l, 16, 0, 0);
}

#define EPI 2
__device__ __forceinline__ void st_out(float* q, float y) {
    asm volatile("global_store_dword %0, %1, off sc0 sc1" :: "v"(q), "v"(y) : "memory");
}

__device__ __forceinline__ v8i cat8(v4i a, v4i b) { return __builtin_shufflevector(a, b, 0, 1, 2, 3, 4, 5, 6, 7); }

#define MFMA(a, b, c) __builtin_amdgcn_mfma_scale_f32_32x32x64_f8f6f4(a, b, c, 0, 0, 0, 0x7f7f7f7f, 0, 0x7f7f7f7f)

__global__ void __launch_bounds__(NTHREADS, 2) fused_kernel(Params p) {
    extern __shared__ __attribute__((aligned(16))) u8 smem[];
    const int tid = threadIdx.x;
    const int lane = tid & 63;
    const int wave = __builtin_amdgcn_readfirstlane(tid >> 6);
    const int b = blockIdx.x;
    const int xcd = b & 7;
    const int idx = b >> 3;
    const int rank = p.rank;
    u8* myws = p.ws[rank];

    u32* cntp = (u32*)(myws + WS_CNT) + b;
    const u32 call = __hip_atomic_load(cntp, __ATOMIC_RELAXED, __HIP_MEMORY_SCOPE_AGENT) + 1u;
    const u32 parity = call & 1u;
    STAMP(0);

    if (idx < 2) {
        const int j = idx * 8 + xcd;
        const int gw = j * 8 + wave;
        const int row0 = 8 * (gw >> 1) + (lane >> 3);
        const int kh = gw & 1;
        const int kb = (lane & 7) * 16;
        v4i v[NCH];
#pragma unroll
        for (int c = 0; c < NCH; ++c) v[c] = *(const v4i*)(p.x + (size_t)row0 * KD + c * KCH + kh * BK + kb);
        const size_t doff = (size_t)kh * (MS * BK) + (size_t)row0 * BK + kb;
        constexpr size_t DSTEP = (size_t)SPC * MS * BK;
        const size_t dbase = WS_STAGE + (size_t)parity * SLOT_BYTES + (size_t)rank * SHARD_BYTES;
        const int fp = (rank + lane) & 7;
        u32* fdst = (u32*)(p.ws[fp] + WS_FLAG) + rank * (NCH * 16) + j;
        u8* dp[NR - 1];
#pragma unroll
        for (int q = 1; q < NR; ++q) dp[q - 1] = p.ws[(rank + q) & 7] + dbase + doff;
        __builtin_amdgcn_s_waitcnt(0x0F70);
#define EARLY 8
        const bool early = (wave < EARLY);
#pragma unroll
        for (int c = 0; c < NCH + 1; ++c) {
            if (c < NCH && (early || c == 0)) {
#pragma unroll
                for (int q = 1; q < NR; ++q) st16_sys(dp[q - 1] + c * DSTEP, v[c]);
            }
            if (c >= 1) {
                if (c < NCH && early) { WAITV(7); } else { WAITV(0); }
                __builtin_amdgcn_s_barrier();
                if (wave == 0 && lane >= 1 && lane < NR)
                    __hip_atomic_store(fdst + (c - 1) * 16, call, __ATOMIC_RELAXED, __HIP_MEMORY_SCOPE_SYSTEM);
                STAMP(c);
                if (c < NCH && !early) {
#pragma unroll
                    for (int q = 1; q < NR; ++q) st16_sys(dp[q - 1] + c * DSTEP, v[c]);
                }
            }
        }
        if (tid == 0) __hip_atomic_store(cntp, call, __ATOMIC_RELAXED, __HIP_MEMORY_SCOPE_AGENT);
        return;
    }

    if (tid == 0) __hip_atomic_store(cntp, call, __ATOMIC_RELAXED, __HIP_MEMORY_SCOPE_AGENT);

#define DMAFIRST 0
#define MAPG 8
    const int L = xcd * 30 + (idx - 2);
    const int grp = L / (15 * MAPG);
    const int e = (L % (15 * MAPG)) / MAPG;
    const int cb = grp * MAPG + (L % MAPG);
    const bool local = (e == 14);
    const bool extra = (e < 8);
    const int src = local ? rank : ((rank + 1 + (e >> 1)) & 7);
    const int rb = local ? 0 : (e & 1);

    u32* tmo = (u32*)(myws + WS_TMO);
    const u32* flags = (const u32*)(myws + WS_FLAG);

    const int wm = wave >> 1, wn = wave & 1;
    const int sr = lane >> 3, sc = lane & 7;
    const int h = lane >> 5;
    const u32 lds0 = (u32)(size_t)(__attribute__((address_space(3))) u8*)smem;
    const u32 ctl_addr = lds0 + LDS_CTL;
    const int arow = wm * 64 + (lane & 31);
    const u32 a0 = lds0 + (u32)arow * 128 + (u32)(((2 * h) ^ ((arow >> 1) & 7)) * 16);
    const int gq = (lane & 15) >> 1, gp = lane & 1, gg1 = (lane >> 4) & 1;
    const int jsw = wm & 1;
    const u32 b0 = (lds0 + STAGE_A + (u32)(32 * h + gq) * 128 +
                    (u32)(((wn * 4 + gg1) ^ (((gq >> 1) & 3) << 1)) * 16 + 8 * gp)) ^ (u32)(jsw << 5);

    const u8* abase = local ? p.x
                            : (myws + WS_STAGE + (size_t)parity * SLOT_BYTES + (size_t)src * SHARD_BYTES + (size_t)rb * 256 * BK);
    const u32 arst = local ? KD : BK, astep = local ? BK : MS * BK;
    const u8* bbase = p.w + (size_t)rank * NS + (size_t)cb * BN;

    const float scale = p.sx[0] * p.sw[0];

    if (extra) {
        const u8* g1; u32 v1a, v1b, g1step, g1piece, l1off;
        if (wave < 4) {
            g1 = bbase + (size_t)(wave * 32) * ND;
            v1a = (u32)sr * ND + (u32)((sc ^ (((sr >> 1) & 3) << 1)) * 16);
            v1b = v1a + 8 * ND;
            g1step = BK * ND; g1piece = 16 * ND; l1off = (u32)wave * 4096;
        } else {
            g1 = p.x + (size_t)(256 + 32 * e) * KD;
            v1a = (u32)sr * KD + (u32)((sc ^ (sr >> 1)) * 16);
            v1b = (u32)(8 + sr) * KD + (u32)((sc ^ (4 + (sr >> 1))) * 16);
            g1step = BK; g1piece = 16 * KD; l1off = (u32)STAGE_B;
        }
        auto stage1 = [&](int t, u32 sl) {
            if (wave <= 4) {
                u8* sd = smem + sl + l1off;
                const u8* g = g1 + (size_t)((u32)t * g1step);
                glds16(g + v1a, sd);
                glds16(g + v1b, sd + 1024);
                g += g1piece;
                glds16(g + v1a, sd + 2048);
                glds16(g + v1b, sd + 3072);
            }
        };
#pragma unroll
        for (int t = 0; t < NSTAGE1 - 1; ++t) stage1(t, (u32)(t * STAGE1));
        v16f accx;
#pragma unroll
        for (int r = 0; r < 16; ++r) accx[r] = 0.f;
        const u32 bx = b0 - (u32)STAGE_A;
        const u32 xx = a0 - (u32)wm * 8192 + (u32)STAGE_B;
        u32 sl = 0;
        for (int t = 0; t < NT; ++t) {
            if (t + NSTAGE1 - 2 < NT) { WAITV(20); } else { WAITV(0); }
            __builtin_amdgcn_s_barrier();
            const u32 slp = (sl == 0) ? (u32)((NSTAGE1 - 1) * STAGE1) : sl - (u32)STAGE1;
            if (t + NSTAGE1 - 1 < NT) stage1(t + NSTAGE1 - 1, slp);
            if (wave >= 4) {
                v4i xl[2][2]; v2i bl[2][4];
                const u32 bb = bx + sl;
                asm volatile("ds_read_b128 %0, %1" : "=v"(xl[0][0]) : "v"(xx + sl));
                asm volatile("ds_read_b128 %0, %1" : "=v"(xl[0][1]) : "v"((xx ^ 16u) + sl));
                asm volatile("ds_read_b64_tr_b8 %0, %1 offset:0" : "=v"(bl[0][0]) : "v"(bb));
                asm volatile("ds_read_b64_tr_b8 %0, %1 offset:1024" : "=v"(bl[0][1]) : "v"(bb));
                asm volatile("ds_read_b64_tr_b8 %0, %1 offset:2048" : "=v"(bl[0][2]) : "v"(bb));
                asm volatile("ds_read_b64_tr_b8 %0, %1 offset:3072" : "=v"(bl[0][3]) : "v"(bb));
                asm volatile("ds_read_b128 %0, %1" : "=v"(xl[1][0]) : "v"((xx ^ 64u) + sl));
                asm volatile("ds_read_b128 %0, %1" : "=v"(xl[1][1]) : "v"((xx ^ 80u) + sl));
                asm volatile("ds_read_b64_tr_b8 %0, %1 offset:8192" : "=v"(bl[1][0]) : "v"(bb));
                asm volatile("ds_read_b64_tr_b8 %0, %1 offset:9216" : "=v"(bl[1][1]) : "v"(bb));
                asm volatile("ds_read_b64_tr_b8 %0, %1 offset:10240" : "=v"(bl[1][2]) : "v"(bb));
                asm volatile("ds_read_b64_tr_b8 %0, %1 offset:11264" : "=v"(bl[1][3]) : "v"(bb));
                asm volatile("s_waitcnt lgkmcnt(0)"
                             : "+v"(xl[0][0]), "+v"(xl[0][1]), "+v"(xl[1][0]), "+v"(xl[1][1]),
                               "+v"(bl[0][0]), "+v"(bl[0][1]), "+v"(bl[0][2]), "+v"(bl[0][3]),
                               "+v"(bl[1][0]), "+v"(bl[1][1]), "+v"(bl[1][2]), "+v"(bl[1][3]));
#pragma unroll
                for (int kk = 0; kk < 2; ++kk)
                    accx = MFMA(cat8(xl[kk][0], xl[kk][1]),
                                cat8(__builtin_shufflevector(bl[kk][0], bl[kk][1], 0, 1, 2, 3),
                                     __builtin_shufflevector(bl[kk][2], bl[kk][3], 0, 1, 2, 3)), accx);
            }
            sl = (sl == (u32)((NSTAGE1 - 1) * STAGE1)) ? 0u : sl + (u32)STAGE1;
        }
        if (wave >= 4) {
            float* obase = p.out + (size_t)(rank * MS + 256 + 32 * e + 4 * h) * NS + cb * BN + wn * 64 + jsw * 32 + (lane & 31);
#pragma unroll
            for (int r = 0; r < 16; ++r) {
                float y = accx[r] * scale;
                y = y > 0.f ? y : 0.f;
                st_out(&obase[(size_t)((r & 3) + 8 * (r >> 2)) * NS], y);
            }
        }
        __builtin_amdgcn_s_barrier();
        STAMP(22);
    }

    const u8* gbase;
    u32 vo0, vo1, gstep, gpiece, ldsoff;
    if (wave < 4) {
        gbase = abase + (size_t)(wave * 64) * arst;
        vo0 = (u32)sr * arst + (u32)((sc ^ (sr >> 1)) * 16);
        vo1 = (u32)(8 + sr) * arst + (u32)((sc ^ (4 + (sr >> 1))) * 16);
        gstep = astep; gpiece = 16 * arst; ldsoff = (u32)wave * 8192;
    } else {
        const int w4 = (wave - 4) & 1;
        gbase = bbase + (size_t)(w4 * 64) * ND;
        vo0 = (u32)sr * ND + (u32)((sc ^ (((sr >> 1) & 3) << 1)) * 16);
        vo1 = vo0 + 8 * ND;
        gstep = BK * ND; gpiece = 16 * ND; ldsoff = (u32)STAGE_A + (u32)w4 * 8192;
    }
    const bool stager = wave < 6;
    int nst = 0;
    u32 nsl = ldsoff;
    auto stage_next = [&]() {
        if (stager) {
            u8* sd = smem + nsl;
            const u8* g = gbase + (size_t)((u32)nst * gstep);
#pragma unroll
            for (int i = 0; i < 4; ++i) {
                glds16(g + vo0, sd + i * 2048);
                glds16(g + vo1, sd + i * 2048 + 1024);
                g += gpiece;
            }
        }
        ++nst;
        nsl = (nsl >= (u32)((NSTAGE - 1) * STAGE)) ? nsl - (u32)((NSTAGE - 1) * STAGE) : nsl + (u32)STAGE;
    };

    const u8* ftab = (const u8*)(flags + src * (NCH * 16));
    auto ok4 = [&](v4i f) -> bool {
        return ((int)((u32)f[0] - call) | (int)((u32)f[1] - call) | (int)((u32)f[2] - call) | (int)((u32)f[3] - call)) >= 0;
    };
    bool bad = false;
    auto poll = [&](int need) -> int {
        unsigned long long t0 = 0;
        const int s0 = min(need - 1, NCH - 4);
        const u8* fp = ftab + s0 * 64 + (lane & 15) * 16;
        while (true) {
            v4i f;
            asm volatile("global_load_dwordx4 %0, %1, off sc0 sc1\n\ts_waitcnt vmcnt(0)" : "=v"(f) : "v"(fp) : "memory");
            const unsigned long long m = __builtin_amdgcn_ballot_w64(ok4(f)) | ~0xffffull;
            const int r = s0 + (__builtin_ctzll(~m | (1ull << 16)) >> 2);
            if (r >= need) return r;
            if (t0 == 0) t0 = __builtin_amdgcn_s_memrealtime();
            if (__hip_atomic_load(tmo, __ATOMIC_RELAXED, __HIP_MEMORY_SCOPE_AGENT) == call ||
                __builtin_amdgcn_s_memrealtime() - t0 > DEADLINE_TICKS) {
                __hip_atomic_store(tmo, call, __ATOMIC_RELAXED, __HIP_MEMORY_SCOPE_AGENT);
                return NCH | 256;
            }
            __builtin_amdgcn_s_sleep(1);
        }
    };
    auto sync_ready = [&](int need) -> int {
        if (wave == 7) {
            const int r = poll(need);
            asm volatile("ds_write_b32 %0, %1\n\ts_waitcnt lgkmcnt(0)" :: "v"(ctl_addr), "v"(r) : "memory");
        }
        __builtin_amdgcn_s_barrier();
        int rv;
        asm volatile("ds_read_b32 %0, %1\n\ts_waitcnt lgkmcnt(0)" : "=v"(rv) : "v"(ctl_addr) : "memory");
        rv = __builtin_amdgcn_readfirstlane(rv);
        if (rv & 256) bad = true;
        return (rv & 255) * SPC;
    };
    const u32 mir_addr = lds0 + LDS_MIR + (u32)lane * 16;
    auto flag_dma = [&](int rdy_steps) {
        const int s0 = min(rdy_steps / SPC, NCH - 4);
        if (lane < 16)
            __builtin_amdgcn_global_load_lds((const u32*)(ftab + s0 * 64 + lane * 16), (u32*)(smem + LDS_MIR + s0 * 64), 16, 0, 17);
    };
    const bool awave = (wave < 4);
    auto wait_landed = [&](int cnt) {
        if (stager) {
            if (cnt >= 2) { WAITV(16); } else if (cnt == 1) { WAITV(8); } else { WAITV(0); }
        }
    };

    if (!awave) { stage_next(); stage_next(); stage_next(); }
    if (wave == 7) {
        const v4i nr = {(int)(call - 1u), (int)(call - 1u), (int)(call - 1u), (int)(call - 1u)};
        asm volatile("ds_write_b128 %0, %1\n\ts_waitcnt lgkmcnt(0)" :: "v"(mir_addr), "v"(nr) : "memory");
    }
    int rdy = local ? NT : sync_ready(1);
    STAMP(1);
    if (awave) {
        const int lim = min(3, rdy);
        while (nst < lim) stage_next();
    }
    if (wave == 7 && rdy < NT) flag_dma(rdy);
    wait_landed(nst - 1);
    __builtin_amdgcn_s_barrier();

    struct Frag { v4i a[2][2]; v2i b[2][4]; };
#define SCHED_FENCE() __builtin_amdgcn_sched_barrier(0)
#define FRAG_OPS(f) "+v"(f.a[0][0]), "+v"(f.a[0][1]), "+v"(f.a[1][0]), "+v"(f.a[1][1]), \
                    "+v"(f.b[0][0]), "+v"(f.b[0][1]), "+v"(f.b[0][2]), "+v"(f.b[0][3]), \
                    "+v"(f.b[1][0]), "+v"(f.b[1][1]), "+v"(f.b[1][2]), "+v"(f.b[1][3])
#define BFRAG(f, j) cat8(__builtin_shufflevector(f.b[j][0], f.b[j][1], 0, 1, 2, 3), \
                         __builtin_shufflevector(f.b[j][2], f.b[j][3], 0, 1, 2, 3))

    {
        auto read_frag = [&](Frag& f, u32 sbase, auto kk_tag) {
            constexpr int kk = decltype(kk_tag)::value;
            const u32 aa0 = (a0 ^ (u32)(kk << 6)) + sbase;
            const u32 aa1 = (a0 ^ (u32)((kk << 6) | 16)) + sbase;
            const u32 bb0 = b0 + sbase;
            const u32 bb1 = (b0 ^ 32u) + sbase;
            asm volatile("ds_read_b128 %0, %1 offset:0" : "=v"(f.a[0][0]) : "v"(aa0));
            asm volatile("ds_read_b128 %0, %1 offset:0" : "=v"(f.a[0][1]) : "v"(aa1));
            if (kk == 0) {
                asm volatile("ds_read_b64_tr_b8 %0, %1 offset:0" : "=v"(f.b[0][0]) : "v"(bb0));
                asm volatile("ds_read_b64_tr_b8 %0, %1 offset:1024" : "=v"(f.b[0][1]) : "v"(bb0));
                asm volatile("ds_read_b64_tr_b8 %0, %1 offset:2048" : "=v"(f.b[0][2]) : "v"(bb0));
                asm volatile("ds_read_b64_tr_b8 %0, %1 offset:3072" : "=v"(f.b[0][3]) : "v"(bb0));
                asm volatile("ds_read_b64_tr_b8 %0, %1 offset:0" : "=v"(f.b[1][0]) : "v"(bb1));
                asm volatile("ds_read_b64_tr_b8 %0, %1 offset:1024" : "=v"(f.b[1][1]) : "v"(bb1));
                asm volatile("ds_read_b64_tr_b8 %0, %1 offset:2048" : "=v"(f.b[1][2]) : "v"(bb1));
                asm volatile("ds_read_b64_tr_b8 %0, %1 offset:3072" : "=v"(f.b[1][3]) : "v"(bb1));
            } else {
                asm volatile("ds_read_b64_tr_b8 %0, %1 offset:8192" : "=v"(f.b[0][0]) : "v"(bb0));
                asm volatile("ds_read_b64_tr_b8 %0, %1 offset:9216" : "=v"(f.b[0][1]) : "v"(bb0));
                asm volatile("ds_read_b64_tr_b8 %0, %1 offset:10240" : "=v"(f.b[0][2]) : "v"(bb0));
                asm volatile("ds_read_b64_tr_b8 %0, %1 offset:11264" : "=v"(f.b[0][3]) : "v"(bb0));
                asm volatile("ds_read_b64_tr_b8 %0, %1 offset:8192" : "=v"(f.b[1][0]) : "v"(bb1));
                asm volatile("ds_read_b64_tr_b8 %0, %1 offset:9216" : "=v"(f.b[1][1]) : "v"(bb1));
                asm volatile("ds_read_b64_tr_b8 %0, %1 offset:10240" : "=v"(f.b[1][2]) : "v"(bb1));
                asm volatile("ds_read_b64_tr_b8 %0, %1 offset:11264" : "=v"(f.b[1][3]) : "v"(bb1));
            }
            asm volatile("ds_read_b128 %0, %1 offset:4096" : "=v"(f.a[1][0]) : "v"(aa0));
            asm volatile("ds_read_b128 %0, %1 offset:4096" : "=v"(f.a[1][1]) : "v"(aa1));
        };
        using K0 = std::integral_constant<int, 0>;
        using K1 = std::integral_constant<int, 1>;

        v16f acc[2][2];
#pragma unroll
        for (int r = 0; r < 16; ++r) {
            acc[0][0][r] = 0.f; acc[0][1][r] = 0.f; acc[1][0][r] = 0.f; acc[1][1][r] = 0.f;
        }
        Frag f0, f1;
        u32 sb = 0;
        read_frag(f0, sb, K0{});

        for (int t = 0; t < NT; ++t) {
            const u32 sbn = (sb == (u32)((NSTAGE - 1) * STAGE)) ? 0u : sb + (u32)STAGE;
            const bool chk = (rdy < NT);
            read_frag(f1, sb, K1{});
            v4i fv = {0, 0, 0, 0};
            if (wave == 7 && chk) asm volatile("ds_read_b128 %0, %1" : "=v"(fv) : "v"(mir_addr) : "memory");
            asm volatile("s_waitcnt lgkmcnt(12)" : FRAG_OPS(f0));
            SCHED_FENCE();
            {
                const v8i a_0 = cat8(f0.a[0][0], f0.a[0][1]), a_1 = cat8(f0.a[1][0], f0.a[1][1]);
                const v8i b_0 = BFRAG(f0, 0), b_1 = BFRAG(f0, 1);
                acc[0][0] = MFMA(a_0, b_0, acc[0][0]);
                acc[0][1] = MFMA(a_0, b_1, acc[0][1]);
                acc[1][0] = MFMA(a_1, b_0, acc[1][0]);
                acc[1][1] = MFMA(a_1, b_1, acc[1][1]);
            }
            asm volatile("" : "+v"(acc[0][0]), "+v"(acc[0][1]), "+v"(acc[1][0]), "+v"(acc[1][1]));
            SCHED_FENCE();
            asm volatile("s_waitcnt lgkmcnt(0)" : FRAG_OPS(f1), "+v"(fv));
            const u32 snap_addr = ctl_addr + 8 + (u32)(t & 1) * 4;
            int rv = 0;
            if (t + 1 < NT) {
                if (chk && t + 1 >= rdy) {
                    rdy = sync_ready((t + 1) / SPC + 1);
                    if (awave) {
                        const int lim = min(t + 4, rdy);
                        while (nst < lim) stage_next();
                    }
                }
                if (wave == 7 && chk) {
                    const unsigned long long m = __builtin_amdgcn_ballot_w64(ok4(fv) || (lane >> 2) * SPC < rdy);
                    const int r = (m == ~0ull ? NCH : (__builtin_ctzll(~m) >> 2)) * SPC;
                    asm volatile("ds_write_b32 %0, %1\n\ts_waitcnt lgkmcnt(0)" :: "v"(snap_addr), "v"(r) : "memory");
                }
                wait_landed(nst - (t + 2));
                __builtin_amdgcn_s_barrier();
            }
            read_frag(f0, sbn, K0{});
            if (chk) asm volatile("ds_read_b32 %0, %1" : "=v"(rv) : "v"(snap_addr) : "memory");
            SCHED_FENCE();
            const v8i a_0 = cat8(f1.a[0][0], f1.a[0][1]), a_1 = cat8(f1.a[1][0], f1.a[1][1]);
            const v8i b_0 = BFRAG(f1, 0), b_1 = BFRAG(f1, 1);
            acc[0][0] = MFMA(a_0, b_0, acc[0][0]);
            acc[0][1] = MFMA(a_0, b_1, acc[0][1]);
            asm volatile("" : "+v"(acc[0][0]), "+v"(acc[0][1]));
            SCHED_FENCE();
            if (chk) {
                asm volatile("s_waitcnt lgkmcnt(0)" : "+v"(rv));
                rdy = __builtin_amdgcn_readfirstlane(rv);
                if (wave == 7 && rdy < NT) flag_dma(rdy);
            }
            {
                const int lim = min(t + 4, awave ? rdy : NT);
                while (nst < lim) stage_next();
            }
            SCHED_FENCE();
            acc[1][0] = MFMA(a_1, b_0, acc[1][0]);
            acc[1][1] = MFMA(a_1, b_1, acc[1][1]);
            asm volatile("" : "+v"(acc[1][0]), "+v"(acc[1][1]));
            SCHED_FENCE();
            sb = sbn;
        }
        asm volatile("s_waitcnt lgkmcnt(0)" ::: "memory");

        STAMP(20);
        {
            float* obase = p.out + (size_t)(src * MS + rb * 256 + wm * 64 + 4 * h) * NS + cb * BN + wn * 64 + (lane & 31);
#pragma unroll
            for (int i = 0; i < 2; ++i)
#pragma unroll
                for (int j = 0; j < 2; ++j)
#pragma unroll
                    for (int r = 0; r < 16; ++r) {
                        float y = acc[i][j][r] * scale;
                        y = y > 0.f ? y : 0.f;
                        if (bad) y = __builtin_nanf("");
                        st_out(&obase[(size_t)(i * 32 + (r & 3) + 8 * (r >> 2)) * NS + (j ^ jsw) * 32], y);
                    }
        }
    }
}

}

extern "C" void kernel_launch(int rank, int nranks, ncclComm_t comm,
                              hipStream_t stream,
                              void* const* d_in, const int* in_sizes, int n_in,
                              void* d_out, int out_size,
                              const int* in_dtypes, int out_dtype,
                              void* const* d_out_all,
                              void* const* d_ws_all, size_t ws_bytes) {
    if (nranks != NR || ws_bytes < WS_NEED) {
        fprintf(stderr, "unsupported configuration\n");
        return;
    }
    static bool attr_set = false;
    if (!attr_set) {
        HIP_CHECK(hipFuncSetAttribute((const void*)fused_kernel,
                                      hipFuncAttributeMaxDynamicSharedMemorySize, LDS_BYTES));
        attr_set = true;
    }
    Params p;
    p.x = (const u8*)d_in[0];
    p.w = (const u8*)d_in[1];
    p.sx = (const float*)d_in[2];
    p.sw = (const float*)d_in[3];
    p.out = (float*)d_out;
    for (int i = 0; i < NR; ++i) p.ws[i] = (u8*)d_ws_all[i];
    p.rank = rank;
    hipLaunchKernelGGL(fused_kernel, dim3(NWG), dim3(NTHREADS), LDS_BYTES, stream, p);
    HIP_CHECK(hipGetLastError());
}
```

```cpp
#include <hip/hip_runtime.h>
#include <rccl/rccl.h>
#include <cstdio>
#include <type_traits>

#define HIP_CHECK(expr) do { hipError_t _e = (expr); if (_e != hipSuccess) { \
    fprintf(stderr, "HIP error %s at %s:%d\n", hipGetErrorString(_e), __FILE__, __LINE__); \
    return; } } while (0)

namespace {

typedef int v8i __attribute__((ext_vector_type(8)));
typedef int v4i __attribute__((ext_vector_type(4)));
typedef int v2i __attribute__((ext_vector_type(2)));
typedef float v16f __attribute__((ext_vector_type(16)));
typedef unsigned int u32;
typedef unsigned char u8;

constexpr int NR = 8;
constexpr int KD = 4096;
constexpr int ND = 16384;
constexpr int MS = 512;
constexpr int NS = 2048;
constexpr int BM = 256, BN = 128, BK = 128;
constexpr int NT = KD / BK;
constexpr int NTHREADS = 768;
constexpr int NWG = 256;
constexpr int KCH = 256;
constexpr int NCH = KD / KCH;
constexpr int SPC = KCH / BK;
constexpr int STAGE_A = BM * BK;
constexpr int STAGE_B = BK * BN;
constexpr int STAGE_X = 32 * BK;
constexpr int STAGE = STAGE_A + STAGE_B + STAGE_X;
constexpr int NSTAGE = 3;
constexpr int LDS_CTL = NSTAGE * STAGE;
constexpr int LDS_BYTES = LDS_CTL + 128;
constexpr int NPW = 16;

constexpr size_t WS_CNT = 0;
constexpr size_t WS_TMO = 4096;
constexpr size_t WS_FLAG = 8192;
constexpr size_t WS_STAGE = 1ull << 20;
constexpr size_t SHARD_BYTES = (size_t)MS * KD;
constexpr size_t SLOT_BYTES = (size_t)NR * SHARD_BYTES;
constexpr size_t WS_NEED = WS_STAGE + 2 * SLOT_BYTES;

constexpr unsigned long long DEADLINE_TICKS = 40000000ull;

struct Params {
    const u8* x;
    const u8* w;
    const float* sx;
    const float* sw;
    float* out;
    u8* ws[NR];
    int rank;
};

__device__ __forceinline__ void st16_sys(const u8* sbase, u32 voff, v4i v) {
    asm volatile("global_store_dwordx4 %0, %1, %2 sc0 sc1\n\ts_nop 1" :: "v"(voff), "v"(v), "s"(sbase) : "memory");
}

#define STAMP(i) do { } while (0)
#define WAITV(n) asm volatile("s_waitcnt vmcnt(" #n ")" ::: "memory")

__device__ __forceinline__ void glds16(const u8* g, u8* l) {
    __builtin_amdgcn_global_load_lds((const u32*)g, (u32*)l, 16, 0, 0);
}

#define EPI 2
__device__ __forceinline__ void st_out(float* q, float y) {
    asm volatile("global_store_dword %0, %1, off sc0 sc1" :: "v"(q), "v"(y) : "memory");
}

__device__ __forceinline__ v8i cat8(v4i a, v4i b) { return __builtin_shufflevector(a, b, 0, 1, 2, 3, 4, 5, 6, 7); }

#define MFMA(a, b, c) __builtin_amdgcn_mfma_scale_f32_32x32x64_f8f6f4(a, b, c, 0, 0, 0, 0x7f7f7f7f, 0, 0x7f7f7f7f)

__global__ void __launch_bounds__(NTHREADS, 3) fused_kernel(Params p) {
    extern __shared__ __attribute__((aligned(16))) u8 smem[];
    const int tid = threadIdx.x;
    const int lane = tid & 63;
    const int wave = __builtin_amdgcn_readfirstlane(tid >> 6);
    const int b = blockIdx.x;
    const int xcd = b & 7;
    const int idx = b >> 3;
    const int rank = p.rank;
    u8* myws = p.ws[rank];

    const u32 lds0 = (u32)(size_t)(__attribute__((address_space(3))) u8*)smem;
    const u32 ctl = lds0 + LDS_CTL;
    const int sr = lane >> 3, sc = lane & 7;
    const u32* cntp = (const u32*)(myws + WS_CNT) + b;

    if (idx < 2) {
        if (wave >= 8) return;
        const int j = idx * 8 + xcd;
        const int gw = j * 8 + wave;
        const int row0 = 8 * (gw >> 1) + sr;
        const int kh = gw & 1;
        const u8* xs = p.x + (size_t)row0 * KD + kh * BK + sc * 16;
        v4i v[NCH];
        u32 cnt;
        asm volatile("global_load_dword %0, %1, off sc1" : "=v"(cnt) : "v"(cntp) : "memory");
#pragma unroll
        for (int c = 0; c < NCH; ++c)
            asm volatile("global_load_dwordx4 %0, %1, off offset:%2" : "=v"(v[c]) : "v"(xs), "n"(c * KCH) : "memory");
        const u8* dp[NR - 1];
        const u8* fb = nullptr;
#pragma unroll
        for (int q = 1; q < NR; ++q) dp[q - 1] = p.ws[(rank + q) & 7];
        asm volatile("" : "+s"(dp[0]), "+s"(dp[1]), "+s"(dp[2]), "+s"(dp[3]), "+s"(dp[4]), "+s"(dp[5]), "+s"(dp[6]));
#pragma unroll
        for (int q = 1; q < NR; ++q) fb = (lane == q) ? dp[q - 1] : fb;
        const u32* fdst = (const u32*)(fb + WS_FLAG) + rank * (NCH * NPW) + j;
        asm volatile("s_waitcnt vmcnt(14)" : "+v"(cnt), "+v"(v[0]), "+v"(v[1]) :: "memory");
        const u32 call = (u32)__builtin_amdgcn_readfirstlane((int)cnt) + 1u;
        const size_t dbase = WS_STAGE + (size_t)(call & 1u) * SLOT_BYTES + (size_t)rank * SHARD_BYTES;
#pragma unroll
        for (int q = 1; q < NR; ++q) dp[q - 1] += dbase;
        const u32 voff = (u32)kh * (MS * BK) + (u32)row0 * BK + (u32)sc * 16;
#pragma unroll
        for (int c = 0; c < NCH + 1; ++c) {
            if (c < NCH) {
#pragma unroll
                for (int q = 1; q < NR; ++q) st16_sys(dp[q - 1], voff + (u32)(c * SPC * MS * BK), v[c]);
            }
            if (c >= 1) {
                if (c == 1) {
                    asm volatile("s_waitcnt vmcnt(7)"
                                 : "+v"(v[2]), "+v"(v[3]), "+v"(v[4]), "+v"(v[5]), "+v"(v[6]), "+v"(v[7]), "+v"(v[8]),
                                   "+v"(v[9]), "+v"(v[10]), "+v"(v[11]), "+v"(v[12]), "+v"(v[13]), "+v"(v[14]), "+v"(v[15])
                                 :: "memory");
                } else if (c < NCH) {
                    if (wave == 0) { WAITV(8); } else { WAITV(7); }
                } else {
                    if (wave == 0) { WAITV(1); } else { WAITV(0); }
                }
                __builtin_amdgcn_s_barrier();
                if (wave == 0 && lane >= 1 && lane < NR)
                    asm volatile("global_store_dword %0, %1, off offset:%2 sc0 sc1" :: "v"(fdst), "v"(call), "n"((c - 1) * NPW * 4) : "memory");
            }
        }
        if (tid == 64) __hip_atomic_store((u32*)cntp, call, __ATOMIC_RELAXED, __HIP_MEMORY_SCOPE_AGENT);
        return;
    }

    const int L = xcd * 30 + (idx - 2);
    const int e = (L % 120) >> 3;
    const int cb = (L / 120) * 8 + (L & 7);
    const bool local = (e == 14);
    const bool extra = (e < 8);
    const int src = local ? rank : ((rank + 1 + (e >> 1)) & 7);
    const int rb = local ? 0 : (e & 1);
    const int h = lane >> 5;
    const int gq = (lane & 15) >> 1, gp = lane & 1, gg1 = (lane >> 4) & 1;

    auto wait_word = [&](u32 addr, int need) {
        while (true) {
            int fv;
            asm volatile("ds_read_b32 %0, %1\n\ts_waitcnt lgkmcnt(0)" : "=v"(fv) : "v"(addr) : "memory");
            if (__builtin_amdgcn_readfirstlane(fv) >= need) break;
            __builtin_amdgcn_s_sleep(1);
        }
    };

    if (wave == 11) {
        __builtin_amdgcn_s_barrier();
        if (!extra) return;
        const float scale = p.sx[0] * p.sw[0];
        const int xr = lane & 31;
        const u32 x0 = lds0 + STAGE_A + STAGE_B + (u32)xr * 128 + (u32)(((2 * h) ^ ((xr >> 1) & 7)) * 16);
        const u32 bs = lds0 + STAGE_A + (u32)(32 * h + gq) * 128 + (u32)((gg1 ^ (((gq >> 1) & 3) << 1)) * 16 + 8 * gp);
        v16f acc[4];
#pragma unroll
        for (int r = 0; r < 16; ++r) { acc[0][r] = 0.f; acc[1][r] = 0.f; acc[2][r] = 0.f; acc[3][r] = 0.f; }
        u32 sb = 0, fa = ctl + 16;
        for (int t = 0; t < NT; ++t) {
            wait_word(fa, t + 1);
            v4i xa[2][2]; v2i bf[2][4][4];
#pragma unroll
            for (int kk = 0; kk < 2; ++kk) {
                asm volatile("ds_read_b128 %0, %1" : "=v"(xa[kk][0]) : "v"((x0 ^ (u32)(kk << 6)) + sb));
                asm volatile("ds_read_b128 %0, %1" : "=v"(xa[kk][1]) : "v"((x0 ^ (u32)((kk << 6) | 16)) + sb));
#pragma unroll
                for (int jj = 0; jj < 4; ++jj) {
                    const u32 bb = (bs ^ (u32)(jj << 5)) + sb;
#pragma unroll
                    for (int q = 0; q < 4; ++q)
                        asm volatile("ds_read_b64_tr_b8 %0, %1 offset:%2" : "=v"(bf[kk][jj][q]) : "v"(bb), "n"(kk * 8192 + q * 1024));
                }
            }
#define XOPS(kk) "+v"(xa[kk][0]), "+v"(xa[kk][1]), \
                 "+v"(bf[kk][0][0]), "+v"(bf[kk][0][1]), "+v"(bf[kk][0][2]), "+v"(bf[kk][0][3]), \
                 "+v"(bf[kk][1][0]), "+v"(bf[kk][1][1]), "+v"(bf[kk][1][2]), "+v"(bf[kk][1][3]), \
                 "+v"(bf[kk][2][0]), "+v"(bf[kk][2][1]), "+v"(bf[kk][2][2]), "+v"(bf[kk][2][3]), \
                 "+v"(bf[kk][3][0]), "+v"(bf[kk][3][1]), "+v"(bf[kk][3][2]), "+v"(bf[kk][3][3])
            asm volatile("s_waitcnt lgkmcnt(0)" : XOPS(0));
            asm volatile("" : XOPS(1));
            asm volatile("ds_write_b32 %0, %1" :: "v"(ctl + 32 + 8 * 4), "v"(t + 1) : "memory");
#pragma unroll
            for (int kk = 0; kk < 2; ++kk)
#pragma unroll
                for (int jj = 0; jj < 4; ++jj)
                    acc[jj] = MFMA(cat8(xa[kk][0], xa[kk][1]),
                                   cat8(__builtin_shufflevector(bf[kk][jj][0], bf[kk][jj][1], 0, 1, 2, 3),
                                        __builtin_shufflevector(bf[kk][jj][2], bf[kk][jj][3], 0, 1, 2, 3)), acc[jj]);
            const bool wrap = (sb == (u32)((NSTAGE - 1) * STAGE));
            sb = wrap ? 0u : sb + (u32)STAGE;
            fa = wrap ? ctl + 16 : fa + 4u;
        }
        float* obase = p.out + (size_t)(rank * MS + 256 + 32 * e + 4 * h) * NS + cb * BN + xr;
#pragma unroll
        for (int jj = 0; jj < 4; ++jj)
#pragma unroll
            for (int r = 0; r < 16; ++r) {
                float y = acc[jj][r] * scale;
                y = y > 0.f ? y : 0.f;
                st_out(&obase[(size_t)((r & 3) + 8 * (r >> 2)) * NS + jj * 32], y);
            }
        return;
    }

    if (wave >= 8) {
        __builtin_amdgcn_s_setprio(2);
        if (wave == 8 && lane < 32) {
            const int iv = (lane >= 8 + 8 + (extra ? 1 : 0) && lane < 24) ? 0x7fffffff : 0;
            asm volatile("ds_write_b32 %0, %1" :: "v"(ctl + (u32)lane * 4), "v"(iv) : "memory");
        }
        u32 cnt;
        asm volatile("global_load_dword %0, %1, off sc1\n\ts_waitcnt vmcnt(0) lgkmcnt(0)" : "=v"(cnt) : "v"(cntp) : "memory");
        const u32 call = (u32)__builtin_amdgcn_readfirstlane((int)cnt) + 1u;
        __builtin_amdgcn_s_barrier();
        if (wave == 8 && lane == 0)
            __hip_atomic_store((u32*)cntp, call, __ATOMIC_RELAXED, __HIP_MEMORY_SCOPE_AGENT);

        const u8* abase = local ? p.x
                                : (myws + WS_STAGE + (size_t)(call & 1u) * SLOT_BYTES + (size_t)src * SHARD_BYTES + (size_t)rb * 256 * BK);
        const u32 arst = local ? KD : BK, astep = local ? BK : MS * BK;
        const u8* bbase = p.w + (size_t)rank * NS + (size_t)cb * BN;
        const u8* xbase = p.x + (size_t)(256 + 32 * (e & 7)) * KD;
        const u32 va0 = (u32)sr * arst + (u32)((sc ^ (sr >> 1)) * 16);
        const u32 va1 = (u32)(8 + sr) * arst + (u32)((sc ^ (4 + (sr >> 1))) * 16);
        const u32 vx0 = (u32)sr * KD + (u32)((sc ^ (sr >> 1)) * 16);
        const u32 vx1 = (u32)(8 + sr) * KD + (u32)((sc ^ (4 + (sr >> 1))) * 16);
        const u32 vb0 = (u32)sr * ND + (u32)((sc ^ (((sr >> 1) & 3) << 1)) * 16);
        const u32 vb1 = vb0 + 8 * ND;
        const u8* ftab = myws + WS_FLAG + (size_t)src * (NCH * NPW * 4) + rb * 32 + (lane & 1) * 16;
        u32* tmo = (u32*)(myws + WS_TMO);
        const u32 free_rd = ctl + 32 + (u32)(lane & 15) * 4;
        const int sl = wave - 8;
        u8* sd = smem + sl * STAGE;
        for (int t = sl; t < NT; t += NSTAGE) {
            if (t >= NSTAGE) {
                while (true) {
                    int fv;
                    asm volatile("ds_read_b32 %0, %1\n\ts_waitcnt lgkmcnt(0)" : "=v"(fv) : "v"(free_rd) : "memory");
                    if (__builtin_amdgcn_ballot_w64(fv >= t - (NSTAGE - 1)) == ~0ull) break;
                    __builtin_amdgcn_s_sleep(1);
                }
            }
            {
                const u8* g = bbase + (size_t)((u32)t * (u32)(BK * ND));
#pragma unroll
                for (int i = 0; i < 8; ++i) {
                    glds16(g + vb0, sd + STAGE_A + i * 2048);
                    glds16(g + vb1, sd + STAGE_A + i * 2048 + 1024);
                    g += 16 * ND;
                }
            }
            if (extra) {
                const u8* g = xbase + t * BK;
                glds16(g + vx0, sd + STAGE_A + STAGE_B);
                glds16(g + vx1, sd + STAGE_A + STAGE_B + 1024);
                glds16(g + 16 * KD + vx0, sd + STAGE_A + STAGE_B + 2048);
                glds16(g + 16 * KD + vx1, sd + STAGE_A + STAGE_B + 3072);
            }
            if (!local) {
                unsigned long long t0 = 0;
                const u8* fp = ftab + (t / SPC) * (NPW * 4);
                while (true) {
                    v4i f;
                    asm volatile("global_load_dwordx4 %0, %1, off sc0 sc1\n\ts_waitcnt vmcnt(0)" : "=v"(f) : "v"(fp) : "memory");
                    if (t0 == 0) asm volatile("ds_write_b32 %0, %1" :: "v"(ctl + 16 + (u32)sl * 4), "v"(t + 1) : "memory");
                    const bool ok = ((int)((u32)f[0] - call) | (int)((u32)f[1] - call) | (int)((u32)f[2] - call) | (int)((u32)f[3] - call)) >= 0;
                    if (__builtin_amdgcn_ballot_w64(ok) == ~0ull) break;
                    if (t0 == 0) t0 = __builtin_amdgcn_s_memrealtime();
                    if (__hip_atomic_load(tmo, __ATOMIC_RELAXED, __HIP_MEMORY_SCOPE_AGENT) == call ||
                        __builtin_amdgcn_s_memrealtime() - t0 > DEADLINE_TICKS) {
                        __hip_atomic_store(tmo, call, __ATOMIC_RELAXED, __HIP_MEMORY_SCOPE_AGENT);
                        asm volatile("ds_write_b32 %0, %1" :: "v"(ctl + 96), "v"(1) : "memory");
                        break;
                    }
                    __builtin_amdgcn_s_sleep(1);
                }
            }
            {
                const u8* g = abase + (size_t)((u32)t * astep);
#pragma unroll
                for (int i = 0; i < 16; ++i) {
                    glds16(g + va0, sd + i * 2048);
                    glds16(g + va1, sd + i * 2048 + 1024);
                    g += 16 * arst;
                }
            }
            WAITV(0);
            asm volatile("ds_write_b32 %0, %1" :: "v"(ctl + (u32)sl * 4), "v"(t + 1) : "memory");
        }
        return;
    }

    const int wm = wave >> 1, wn = wave & 1;
    const int arow = wm * 64 + (lane & 31);
    const u32 a0 = lds0 + (u32)arow * 128 + (u32)(((2 * h) ^ ((arow >> 1) & 7)) * 16);
    const int jsw = wm & 1;
    const u32 b0 = (lds0 + STAGE_A + (u32)(32 * h + gq) * 128 +
                    (u32)(((wn * 4 + gg1) ^ (((gq >> 1) & 3) << 1)) * 16 + 8 * gp)) ^ (u32)(jsw << 5);
    const float scale = p.sx[0] * p.sw[0];
    const u32 free_wr = ctl + 32 + (u32)wave * 4;

    struct Frag { v4i a[2][2]; v2i b[2][4]; };
#define SCHED_FENCE() __builtin_amdgcn_sched_barrier(0)
#define FRAG_OPS(f) "+v"(f.a[0][0]), "+v"(f.a[0][1]), "+v"(f.a[1][0]), "+v"(f.a[1][1]), \
                    "+v"(f.b[0][0]), "+v"(f.b[0][1]), "+v"(f.b[0][2]), "+v"(f.b[0][3]), \
                    "+v"(f.b[1][0]), "+v"(f.b[1][1]), "+v"(f.b[1][2]), "+v"(f.b[1][3])
#define BFRAG(f, j) cat8(__builtin_shufflevector(f.b[j][0], f.b[j][1], 0, 1, 2, 3), \
                         __builtin_shufflevector(f.b[j][2], f.b[j][3], 0, 1, 2, 3))
#define MMA4(f) do { \
        SCHED_FENCE(); \
        const v8i a_0 = cat8(f.a[0][0], f.a[0][1]), a_1 = cat8(f.a[1][0], f.a[1][1]); \
        const v8i b_0 = BFRAG(f, 0), b_1 = BFRAG(f, 1); \
        acc[0][0] = MFMA(a_0, b_0, acc[0][0]); \
        acc[0][1] = MFMA(a_0, b_1, acc[0][1]); \
        acc[1][0] = MFMA(a_1, b_0, acc[1][0]); \
        acc[1][1] = MFMA(a_1, b_1, acc[1][1]); \
        asm volatile("" : "+v"(acc[0][0]), "+v"(acc[0][1]), "+v"(acc[1][0]), "+v"(acc[1][1])); \
        SCHED_FENCE(); \
    } while (0)

    auto read_frag = [&](Frag& f, u32 sbase, auto kk_tag) {
        constexpr int kk = decltype(kk_tag)::value;
        const u32 aa0 = (a0 ^ (u32)(kk << 6)) + sbase;
        const u32 aa1 = (a0 ^ (u32)((kk << 6) | 16)) + sbase;
        const u32 bb0 = b0 + sbase;
        const u32 bb1 = (b0 ^ 32u) + sbase;
        asm volatile("ds_read_b128 %0, %1 offset:0" : "=v"(f.a[0][0]) : "v"(aa0));
        asm volatile("ds_read_b128 %0, %1 offset:0" : "=v"(f.a[0][1]) : "v"(aa1));
        asm volatile("ds_read_b64_tr_b8 %0, %1 offset:%2" : "=v"(f.b[0][0]) : "v"(bb0), "n"(kk * 8192));
        asm volatile("ds_read_b64_tr_b8 %0, %1 offset:%2" : "=v"(f.b[0][1]) : "v"(bb0), "n"(kk * 8192 + 1024));
        asm volatile("ds_read_b64_tr_b8 %0, %1 offset:%2" : "=v"(f.b[0][2]) : "v"(bb0), "n"(kk * 8192 + 2048));
        asm volatile("ds_read_b64_tr_b8 %0, %1 offset:%2" : "=v"(f.b[0][3]) : "v"(bb0), "n"(kk * 8192 + 3072));
        asm volatile("ds_read_b64_tr_b8 %0, %1 offset:%2" : "=v"(f.b[1][0]) : "v"(bb1), "n"(kk * 8192));
        asm volatile("ds_read_b64_tr_b8 %0, %1 offset:%2" : "=v"(f.b[1][1]) : "v"(bb1), "n"(kk * 8192 + 1024));
        asm volatile("ds_read_b64_tr_b8 %0, %1 offset:%2" : "=v"(f.b[1][2]) : "v"(bb1), "n"(kk * 8192 + 2048));
        asm volatile("ds_read_b64_tr_b8 %0, %1 offset:%2" : "=v"(f.b[1][3]) : "v"(bb1), "n"(kk * 8192 + 3072));
        asm volatile("ds_read_b128 %0, %1 offset:4096" : "=v"(f.a[1][0]) : "v"(aa0));
        asm volatile("ds_read_b128 %0, %1 offset:4096" : "=v"(f.a[1][1]) : "v"(aa1));
    };
    using K0 = std::integral_constant<int, 0>;
    using K1 = std::integral_constant<int, 1>;

    v16f acc[2][2];
#pragma unroll
    for (int r = 0; r < 16; ++r) {
        acc[0][0][r] = 0.f; acc[0][1][r] = 0.f; acc[1][0][r] = 0.f; acc[1][1][r] = 0.f;
    }
    __builtin_amdgcn_s_barrier();
    Frag f0, f1;
    u32 sb = 0;
    u32 fa = ctl;
    wait_word(fa, 1);
    read_frag(f0, sb, K0{});

    for (int t = 0; t < NT; ++t) {
        const bool wrap = (sb == (u32)((NSTAGE - 1) * STAGE));
        const u32 sbn = wrap ? 0u : sb + (u32)STAGE;
        const u32 fan = wrap ? ctl : fa + 4u;
        read_frag(f1, sb, K1{});
        int fv;
        asm volatile("ds_read_b32 %0, %1" : "=v"(fv) : "v"(fan) : "memory");
        asm volatile("s_waitcnt lgkmcnt(13)" : FRAG_OPS(f0));
        MMA4(f0);
        asm volatile("s_waitcnt lgkmcnt(0)" : FRAG_OPS(f1), "+v"(fv));
        asm volatile("ds_write_b32 %0, %1" :: "v"(free_wr), "v"(t + 1) : "memory");
        const bool ready = (t + 1 == NT) || (__builtin_amdgcn_readfirstlane(fv) >= t + 2);
        if (!ready) {
            MMA4(f1);
            wait_word(fan, t + 2);
        }
        read_frag(f0, sbn, K0{});
        if (ready) MMA4(f1);
        sb = sbn;
        fa = fan;
    }
    int badv;
    asm volatile("ds_read_b32 %0, %1\n\ts_waitcnt lgkmcnt(0)" : "=v"(badv) : "v"(ctl + 96) : "memory");
    const bool bad = badv != 0;

    {
        float* obase = p.out + (size_t)(src * MS + rb * 256 + wm * 64 + 4 * h) * NS + cb * BN + wn * 64 + (lane & 31);
#pragma unroll
        for (int i = 0; i < 2; ++i)
#pragma unroll
            for (int j = 0; j < 2; ++j)
#pragma unroll
                for (int r = 0; r < 16; ++r) {
                    float y = acc[i][j][r] * scale;
                    y = y > 0.f ? y : 0.f;
                    if (bad) y = __builtin_nanf("");
                    st_out(&obase[(size_t)(i * 32 + (r & 3) + 8 * (r >> 2)) * NS + (j ^ jsw) * 32], y);
                }
    }
}

}

extern "C" void kernel_launch(int rank, int nranks, ncclComm_t comm,
                              hipStream_t stream,
                              void* const* d_in, const int* in_sizes, int n_in,
                              void* d_out, int out_size,
                              const int* in_dtypes, int out_dtype,
                              void* const* d_out_all,
                              void* const* d_ws_all, size_t ws_bytes) {
    if (nranks != NR || ws_bytes < WS_NEED) {
        fprintf(stderr, "unsupported configuration\n");
        return;
    }
    static bool attr_set = false;
    if (!attr_set) {
        HIP_CHECK(hipFuncSetAttribute((const void*)fused_kernel,
                                      hipFuncAttributeMaxDynamicSharedMemorySize, LDS_BYTES));
        attr_set = true;
    }
    Params p;
    p.x = (const u8*)d_in[0];
    p.w = (const u8*)d_in[1];
    p.sx = (const float*)d_in[2];
    p.sw = (const float*)d_in[3];
    p.out = (float*)d_out;
    for (int i = 0; i < NR; ++i) p.ws[i] = (u8*)d_ws_all[i];
    p.rank = rank;
    hipLaunchKernelGGL(fused_kernel, dim3(NWG), dim3(NTHREADS), LDS_BYTES, stream, p);
    HIP_CHECK(hipGetLastError());
}
```

```cpp
#include <hip/hip_runtime.h>
#include <rccl/rccl.h>
#include <cstdio>
#include <type_traits>

#define HIP_CHECK(expr) do { hipError_t _e = (expr); if (_e != hipSuccess) { \
    fprintf(stderr, "HIP error %s at %s:%d\n", hipGetErrorString(_e), __FILE__, __LINE__); \
    return; } } while (0)

namespace {

typedef int v8i __attribute__((ext_vector_type(8)));
typedef int v4i __attribute__((ext_vector_type(4)));
typedef int v2i __attribute__((ext_vector_type(2)));
typedef float v16f __attribute__((ext_vector_type(16)));
typedef unsigned int u32;
typedef unsigned char u8;

constexpr int NR = 8;
constexpr int KD = 4096;
constexpr int ND = 16384;
constexpr int MS = 512;
constexpr int NS = 2048;
constexpr int BM = 256, BN = 128, BK = 128;
constexpr int NT = KD / BK;
constexpr int NTHREADS = 768;
constexpr int NWG = 256;
constexpr int KCH = 256;
constexpr int NCH = KD / KCH;
constexpr int SPC = KCH / BK;
constexpr int STAGE_A = BM * BK;
constexpr int STAGE_B = BK * BN;
constexpr int STAGE_X = 32 * BK;
constexpr int STAGE = STAGE_A + STAGE_B + STAGE_X;
constexpr int NSTAGE = 3;
constexpr int LDS_CTL = NSTAGE * STAGE;
constexpr int LDS_BYTES = LDS_CTL + 256;
constexpr int NPW = 16;

constexpr size_t WS_CNT = 0;
constexpr size_t WS_TMO = 4096;
constexpr size_t WS_FLAG = 8192;
constexpr int FROWS = NT;
#define UPK 4
#define UW 4
#define KW 2
constexpr size_t WS_STAGE = 1ull << 20;
constexpr size_t SHARD_BYTES = (size_t)MS * KD;
constexpr size_t SLOT_BYTES = (size_t)NR * SHARD_BYTES;
constexpr size_t WS_NEED = WS_STAGE + 2 * SLOT_BYTES;

constexpr unsigned long long DEADLINE_TICKS = 40000000ull;

struct Params {
    const u8* x;
    const u8* w;
    const float* sx;
    const float* sw;
    float* out;
    u8* ws[NR];
    int rank;
};

__device__ __forceinline__ void st16_sys(const u8* sbase, u32 voff, v4i v) {
    asm volatile("global_store_dwordx4 %0, %1, %2 sc0 sc1\n\ts_nop 1" :: "v"(voff), "v"(v), "s"(sbase) : "memory");
}

#define STAMP(i) do { } while (0)
#define WAITV(n) asm volatile("s_waitcnt vmcnt(" #n ")" ::: "memory")

__device__ __forceinline__ void glds16(const u8* g, u8* l) {
    __builtin_amdgcn_global_load_lds((const u32*)g, (u32*)l, 16, 0, 0);
}

#define EPI 2
__device__ __forceinline__ void st_out(float* q, float y) {
    asm volatile("global_store_dword %0, %1, off sc0 sc1" :: "v"(q), "v"(y) : "memory");
}

__device__ __forceinline__ v8i cat8(v4i a, v4i b) { return __builtin_shufflevector(a, b, 0, 1, 2, 3, 4, 5, 6, 7); }

#define MFMA(a, b, c) __builtin_amdgcn_mfma_scale_f32_32x32x64_f8f6f4(a, b, c, 0, 0, 0, 0x7f7f7f7f, 0, 0x7f7f7f7f)

__global__ void __launch_bounds__(NTHREADS, 3) fused_kernel(Params p) {
    extern __shared__ __attribute__((aligned(16))) u8 smem[];
    const int tid = threadIdx.x;
    const int lane = tid & 63;
    const int wave = __builtin_amdgcn_readfirstlane(tid >> 6);
    const int b = blockIdx.x;
    const int xcd = b & 7;
    const int idx = b >> 3;
    const int rank = p.rank;
    u8* myws = p.ws[rank];

    const u32 lds0 = (u32)(size_t)(__attribute__((address_space(3))) u8*)smem;
    const u32 ctl = lds0 + LDS_CTL;
    const int sr = lane >> 3, sc = lane & 7;
    const u32* cntp = (const u32*)(myws + WS_CNT) + b;

    if (idx < 2) {
        if (wave >= 9) return;
        const int j = (int)((0x642ECA08u >> (4 * xcd)) & 15u) + idx;
        if (wave == 8) {
            u32 cnt;
            asm volatile("global_load_dword %0, %1, off sc1\n\ts_waitcnt vmcnt(0)" : "=v"(cnt) : "v"(cntp) : "memory");
            const u32 call = (u32)__builtin_amdgcn_readfirstlane((int)cnt) + 1u;
            const u8* fb = p.ws[(rank + 1) & 7];
#pragma unroll
            for (int q = 2; q < NR; ++q) fb = (lane == q) ? p.ws[(rank + q) & 7] : fb;
            const u32* fdst = (const u32*)(fb + WS_FLAG) + rank * (FROWS * NPW) + j;
            for (int r = 0; r < NT; ++r) {
                __builtin_amdgcn_s_barrier();
#pragma unroll
                for (int i = 1; i < UPK; ++i) __builtin_amdgcn_s_barrier();
                if (lane >= 1 && lane < NR)
                    asm volatile("global_store_dword %0, %1, off sc0 sc1" :: "v"(fdst), "v"(call) : "memory");
                fdst += NPW;
            }
            if (lane == 0) __hip_atomic_store((u32*)cntp, call, __ATOMIC_RELAXED, __HIP_MEMORY_SCOPE_AGENT);
            return;
        }
        const int gw = j * 8 + wave;
        const int row0 = 8 * (gw >> 1) + sr;
        const int kh = gw & 1;
        const u8* xs = p.x + (size_t)row0 * KD + kh * BK + sc * 16;
        v4i v[NCH];
        u32 cnt;
        asm volatile("global_load_dword %0, %1, off sc1" : "=v"(cnt) : "v"(cntp) : "memory");
#pragma unroll
        for (int c = 0; c < NCH; ++c)
            asm volatile("global_load_dwordx4 %0, %1, off offset:%2" : "=v"(v[c]) : "v"(xs), "n"(c * KCH) : "memory");
        const u8* dp[NR - 1];
#pragma unroll
        for (int q = 1; q < NR; ++q) dp[q - 1] = p.ws[(rank + q) & 7];
        asm volatile("" : "+s"(dp[0]), "+s"(dp[1]), "+s"(dp[2]), "+s"(dp[3]), "+s"(dp[4]), "+s"(dp[5]), "+s"(dp[6]));
        asm volatile("s_waitcnt vmcnt(14)" : "+v"(cnt), "+v"(v[0]), "+v"(v[1]) :: "memory");
        const u32 call = (u32)__builtin_amdgcn_readfirstlane((int)cnt) + 1u;
        const size_t dbase = WS_STAGE + (size_t)(call & 1u) * SLOT_BYTES + (size_t)rank * SHARD_BYTES;
#pragma unroll
        for (int q = 1; q < NR; ++q) dp[q - 1] += dbase;
        const u32 voff = (u32)kh * (MS * BK) + (u32)row0 * BK + (u32)sc * 16;
        const int cls = kh * UPK + ((wave >> 1) & (UPK - 1));
#define IW UW
#define EW 1000
#define EX 8
        int nb = 0;
#pragma unroll
        for (int c = 0; c < NCH; ++c) {
            const int u = 2 * UPK * c + cls;
            int need = u < IW ? 0 : u - (UW - 1);
            if (need > EW) need = (u - (EX - 1)) > EW ? u - (EX - 1) : EW;
            for (; nb < need; ++nb) __builtin_amdgcn_s_barrier();
#pragma unroll
            for (int q = 1; q < NR; ++q) st16_sys(dp[q - 1], voff + (u32)(c * SPC * MS * BK), v[c]);
            for (; nb < u; ++nb) __builtin_amdgcn_s_barrier();
            if (c == 0) {
                asm volatile("s_waitcnt vmcnt(0)"
                     : "+v"(v[2]), "+v"(v[3]), "+v"(v[4]), "+v"(v[5]), "+v"(v[6]), "+v"(v[7]), "+v"(v[8]),
                       "+v"(v[9]), "+v"(v[10]), "+v"(v[11]), "+v"(v[12]), "+v"(v[13]), "+v"(v[14]), "+v"(v[15]) :: "memory");
            } else {
                WAITV(0);
            }
            __builtin_amdgcn_s_barrier();
            ++nb;
        }
        for (; nb < UPK * NT; ++nb) __builtin_amdgcn_s_barrier();
        return;
#pragma unroll
        for (int st = 0; st < NT + KW - 1; ++st) {
            if (st < NT && (st & 1) == kh) {
#pragma unroll
                for (int q = 1; q < NR; ++q) st16_sys(dp[q - 1], voff + (u32)((st >> 1) * SPC * MS * BK), v[st >> 1]);
            }
            const int r = st - (KW - 1);
            if (r >= 0) {
                if ((r & 1) == kh) {
                    constexpr int LATER = (KW - 1) / 2;
                    if (r < 2) {
                        if (LATER == 0) {
                            asm volatile("s_waitcnt vmcnt(0)"
                                 : "+v"(v[2]), "+v"(v[3]), "+v"(v[4]), "+v"(v[5]), "+v"(v[6]), "+v"(v[7]), "+v"(v[8]),
                                   "+v"(v[9]), "+v"(v[10]), "+v"(v[11]), "+v"(v[12]), "+v"(v[13]), "+v"(v[14]), "+v"(v[15]) :: "memory");
                        } else {
                            asm volatile("s_waitcnt vmcnt(7)"
                                 : "+v"(v[2]), "+v"(v[3]), "+v"(v[4]), "+v"(v[5]), "+v"(v[6]), "+v"(v[7]), "+v"(v[8]),
                                   "+v"(v[9]), "+v"(v[10]), "+v"(v[11]), "+v"(v[12]), "+v"(v[13]), "+v"(v[14]), "+v"(v[15]) :: "memory");
                        }
                    } else if (LATER == 1 && r + 2 < NT) {
                        WAITV(7);
                    } else {
                        WAITV(0);
                    }
                }
                __builtin_amdgcn_s_barrier();
            }
        }
        return;
    }

    const int L = xcd * 30 + (idx - 2);
    const int e = (L % 120) >> 3;
    const int cb = (L / 120) * 8 + (L & 7);
    const bool local = (e == 14);
    const bool extra = (e < 8);
    const int src = local ? rank : ((rank + 1 + (e >> 1)) & 7);
    const int rb = local ? 0 : (e & 1);
    const int h = lane >> 5;
    const int gq = (lane & 15) >> 1, gp = lane & 1, gg1 = (lane >> 4) & 1;

    auto wait_word = [&](u32 addr, int need) {
        while (true) {
            int fv;
            asm volatile("ds_read_b32 %0, %1\n\ts_waitcnt lgkmcnt(0)" : "=v"(fv) : "v"(addr) : "memory");
            if (__builtin_amdgcn_readfirstlane(fv) >= need) break;
            __builtin_amdgcn_s_sleep(1);
        }
    };

    if (wave == 11) {
        __builtin_amdgcn_s_barrier();
        if (!extra) return;
        const float scale = p.sx[0] * p.sw[0];
        const int xr = lane & 31;
        const u32 x0 = lds0 + STAGE_A + STAGE_B + (u32)xr * 128 + (u32)(((2 * h) ^ ((xr >> 1) & 7)) * 16);
        const u32 bs = lds0 + STAGE_A + (u32)(32 * h + gq) * 128 + (u32)((gg1 ^ (((gq >> 1) & 3) << 1)) * 16 + 8 * gp);
        v16f acc[4];
#pragma unroll
        for (int r = 0; r < 16; ++r) { acc[0][r] = 0.f; acc[1][r] = 0.f; acc[2][r] = 0.f; acc[3][r] = 0.f; }
        u32 sb = 0, fa = ctl + 48;
        for (int t = 0; t < NT; ++t) {
            wait_word(fa, t + 1);
            v4i xa[2][2]; v2i bf[2][4][4];
#pragma unroll
            for (int kk = 0; kk < 2; ++kk) {
                asm volatile("ds_read_b128 %0, %1" : "=v"(xa[kk][0]) : "v"((x0 ^ (u32)(kk << 6)) + sb));
                asm volatile("ds_read_b128 %0, %1" : "=v"(xa[kk][1]) : "v"((x0 ^ (u32)((kk << 6) | 16)) + sb));
#pragma unroll
                for (int jj = 0; jj < 4; ++jj) {
                    const u32 bb = (bs ^ (u32)(jj << 5)) + sb;
#pragma unroll
                    for (int q = 0; q < 4; ++q)
                        asm volatile("ds_read_b64_tr_b8 %0, %1 offset:%2" : "=v"(bf[kk][jj][q]) : "v"(bb), "n"(kk * 8192 + q * 1024));
                }
            }
#define XOPS(kk) "+v"(xa[kk][0]), "+v"(xa[kk][1]), \
                 "+v"(bf[kk][0][0]), "+v"(bf[kk][0][1]), "+v"(bf[kk][0][2]), "+v"(bf[kk][0][3]), \
                 "+v"(bf[kk][1][0]), "+v"(bf[kk][1][1]), "+v"(bf[kk][1][2]), "+v"(bf[kk][1][3]), \
                 "+v"(bf[kk][2][0]), "+v"(bf[kk][2][1]), "+v"(bf[kk][2][2]), "+v"(bf[kk][2][3]), \
                 "+v"(bf[kk][3][0]), "+v"(bf[kk][3][1]), "+v"(bf[kk][3][2]), "+v"(bf[kk][3][3])
            asm volatile("s_waitcnt lgkmcnt(0)" : XOPS(0));
            asm volatile("" : XOPS(1));
            asm volatile("ds_write_b32 %0, %1" :: "v"(ctl + 64 + 8 * 4), "v"(t + 1) : "memory");
#pragma unroll
            for (int kk = 0; kk < 2; ++kk)
#pragma unroll
                for (int jj = 0; jj < 4; ++jj)
                    acc[jj] = MFMA(cat8(xa[kk][0], xa[kk][1]),
                                   cat8(__builtin_shufflevector(bf[kk][jj][0], bf[kk][jj][1], 0, 1, 2, 3),
                                        __builtin_shufflevector(bf[kk][jj][2], bf[kk][jj][3], 0, 1, 2, 3)), acc[jj]);
            const bool wrap = (sb == (u32)((NSTAGE - 1) * STAGE));
            sb = wrap ? 0u : sb + (u32)STAGE;
            fa = wrap ? ctl + 48 : fa + 4u;
        }
        float* obase = p.out + (size_t)(rank * MS + 256 + 32 * e + 4 * h) * NS + cb * BN + xr;
#pragma unroll
        for (int jj = 0; jj < 4; ++jj)
#pragma unroll
            for (int r = 0; r < 16; ++r) {
                float y = acc[jj][r] * scale;
                y = y > 0.f ? y : 0.f;
                st_out(&obase[(size_t)((r & 3) + 8 * (r >> 2)) * NS + jj * 32], y);
            }
        return;
    }

    if (wave >= 8) {
        __builtin_amdgcn_s_setprio(2);
        if (wave == 8 && lane < 40) {
            const int iv = (lane >= 16 + 8 + (extra ? 1 : 0) && lane < 32) ? 0x7fffffff : 0;
            asm volatile("ds_write_b32 %0, %1" :: "v"(ctl + (u32)lane * 4), "v"(iv) : "memory");
        }
        u32 cnt;
        asm volatile("global_load_dword %0, %1, off sc1\n\ts_waitcnt vmcnt(0) lgkmcnt(0)" : "=v"(cnt) : "v"(cntp) : "memory");
        const u32 call = (u32)__builtin_amdgcn_readfirstlane((int)cnt) + 1u;
        __builtin_amdgcn_s_barrier();
        if (wave == 8 && lane == 0)
            __hip_atomic_store((u32*)cntp, call, __ATOMIC_RELAXED, __HIP_MEMORY_SCOPE_AGENT);

        const u8* abase = local ? p.x
                                : (myws + WS_STAGE + (size_t)(call & 1u) * SLOT_BYTES + (size_t)src * SHARD_BYTES + (size_t)rb * 256 * BK);
        const u32 arst = local ? KD : BK, astep = local ? BK : MS * BK;
        const u8* bbase = p.w + (size_t)rank * NS + (size_t)cb * BN;
        const u8* xbase = p.x + (size_t)(256 + 32 * (e & 7)) * KD;
        const u32 va0 = (u32)sr * arst + (u32)((sc ^ (sr >> 1)) * 16);
        const u32 va1 = (u32)(8 + sr) * arst + (u32)((sc ^ (4 + (sr >> 1))) * 16);
        const u32 vx0 = (u32)sr * KD + (u32)((sc ^ (sr >> 1)) * 16);
        const u32 vx1 = (u32)(8 + sr) * KD + (u32)((sc ^ (4 + (sr >> 1))) * 16);
        const u32 vb0 = (u32)sr * ND + (u32)((sc ^ (((sr >> 1) & 3) << 1)) * 16);
        const u32 vb1 = vb0 + 8 * ND;
        const u8* ftab = myws + WS_FLAG + (size_t)src * (FROWS * NPW * 4) + rb * 32 + (lane & 1) * 16;
        u32* tmo = (u32*)(myws + WS_TMO);
        const u32 free_rd = ctl + 64 + (u32)(lane & 15) * 4;
        const int sl = wave - 8;
        const u32 full_wr = ctl + (u32)sl * 16 + (u32)(lane & 3) * 4;
        u8* sd = smem + sl * STAGE;
        for (int t = sl; t < NT; t += NSTAGE) {
            if (t >= NSTAGE) {
                while (true) {
                    int fv;
                    asm volatile("ds_read_b32 %0, %1\n\ts_waitcnt lgkmcnt(0)" : "=v"(fv) : "v"(free_rd) : "memory");
                    if (__builtin_amdgcn_ballot_w64(fv >= t - (NSTAGE - 1)) == ~0ull) break;
                    __builtin_amdgcn_s_sleep(1);
                }
            }
            {
                const u8* g = bbase + (size_t)((u32)t * (u32)(BK * ND));
#pragma unroll
                for (int i = 0; i < 8; ++i) {
                    glds16(g + vb0, sd + STAGE_A + i * 2048);
                    glds16(g + vb1, sd + STAGE_A + i * 2048 + 1024);
                    g += 16 * ND;
                }
            }
            if (extra) {
                const u8* g = xbase + t * BK;
                glds16(g + vx0, sd + STAGE_A + STAGE_B);
                glds16(g + vx1, sd + STAGE_A + STAGE_B + 1024);
                glds16(g + 16 * KD + vx0, sd + STAGE_A + STAGE_B + 2048);
                glds16(g + 16 * KD + vx1, sd + STAGE_A + STAGE_B + 3072);
            }
            const u8* ga = abase + (size_t)((u32)t * astep);
            auto issue = [&](u32 m) {
#pragma unroll
                for (int gi = 0; gi < 4; ++gi) {
                    if ((m >> gi) & 1u) {
                        const u8* g = ga + (size_t)(gi * 64) * arst;
#pragma unroll
                        for (int i = 0; i < 4; ++i) {
                            glds16(g + va0, sd + gi * 8192 + i * 2048);
                            glds16(g + va1, sd + gi * 8192 + i * 2048 + 1024);
                            g += 16 * arst;
                        }
                    }
                }
            };
            auto publish = [&](u32 m) {
                if ((m >> (lane & 3)) & 1u)
                    asm volatile("ds_write_b32 %0, %1" :: "v"(full_wr), "v"(t + 1) : "memory");
            };
            u32 pend = 15u;
            if (local) {
                issue(15u);
            } else {
                unsigned long long t0 = 0;
                const u8* fp = ftab + t * (NPW * 4);
                u32 done = 0;
                pend = 0;
                bool first = true;
                while (true) {
                    v4i f;
                    asm volatile("global_load_dwordx4 %0, %1, off sc0 sc1\n\ts_waitcnt vmcnt(0)" : "=v"(f) : "v"(fp) : "memory");
                    if (first) asm volatile("ds_write_b32 %0, %1" :: "v"(ctl + 48 + (u32)sl * 4), "v"(t + 1) : "memory");
                    first = false;
                    if (pend) publish(pend);
                    const u32 ma = (u32)__builtin_amdgcn_ballot_w64(((int)((u32)f[0] - call) | (int)((u32)f[1] - call)) >= 0);
                    const u32 mb = (u32)__builtin_amdgcn_ballot_w64(((int)((u32)f[2] - call) | (int)((u32)f[3] - call)) >= 0);
                    pend = ((ma & 1u) | ((mb & 1u) << 1) | ((ma & 2u) << 1) | ((mb & 2u) << 2)) & ~done;
                    if (pend == 0) {
                        if (t0 == 0) t0 = __builtin_amdgcn_s_memrealtime();
                        const unsigned long long dt = __builtin_amdgcn_s_memrealtime() - t0;
                        if (dt > 2000ull &&
                            (__hip_atomic_load(tmo, __ATOMIC_RELAXED, __HIP_MEMORY_SCOPE_AGENT) == call || dt > DEADLINE_TICKS)) {
                            __hip_atomic_store(tmo, call, __ATOMIC_RELAXED, __HIP_MEMORY_SCOPE_AGENT);
                            asm volatile("ds_write_b32 %0, %1" :: "v"(ctl + 128), "v"(1) : "memory");
                            pend = 15u & ~done;
                        } else {
                            __builtin_amdgcn_s_sleep(1);
                            continue;
                        }
                    }
                    issue(pend);
                    done |= pend;
                    if (done == 15u) break;
                }
            }
            WAITV(0);
            publish(pend);
        }
        return;
    }

    const int wm = wave >> 1, wn = wave & 1;
    const int arow = wm * 64 + (lane & 31);
    const u32 a0 = lds0 + (u32)arow * 128 + (u32)(((2 * h) ^ ((arow >> 1) & 7)) * 16);
    const int jsw = wm & 1;
    const u32 b0 = (lds0 + STAGE_A + (u32)(32 * h + gq) * 128 +
                    (u32)(((wn * 4 + gg1) ^ (((gq >> 1) & 3) << 1)) * 16 + 8 * gp)) ^ (u32)(jsw << 5);
    const float scale = p.sx[0] * p.sw[0];
    const u32 free_wr = ctl + 64 + (u32)wave * 4;

    struct Frag { v4i a[2][2]; v2i b[2][4]; };
#define SCHED_FENCE() __builtin_amdgcn_sched_barrier(0)
#define FRAG_OPS(f) "+v"(f.a[0][0]), "+v"(f.a[0][1]), "+v"(f.a[1][0]), "+v"(f.a[1][1]), \
                    "+v"(f.b[0][0]), "+v"(f.b[0][1]), "+v"(f.b[0][2]), "+v"(f.b[0][3]), \
                    "+v"(f.b[1][0]), "+v"(f.b[1][1]), "+v"(f.b[1][2]), "+v"(f.b[1][3])
#define BFRAG(f, j) cat8(__builtin_shufflevector(f.b[j][0], f.b[j][1], 0, 1, 2, 3), \
                         __builtin_shufflevector(f.b[j][2], f.b[j][3], 0, 1, 2, 3))
#define MMA4(f) do { \
        SCHED_FENCE(); \
        const v8i a_0 = cat8(f.a[0][0], f.a[0][1]), a_1 = cat8(f.a[1][0], f.a[1][1]); \
        const v8i b_0 = BFRAG(f, 0), b_1 = BFRAG(f, 1); \
        acc[0][0] = MFMA(a_0, b_0, acc[0][0]); \
        acc[0][1] = MFMA(a_0, b_1, acc[0][1]); \
        acc[1][0] = MFMA(a_1, b_0, acc[1][0]); \
        acc[1][1] = MFMA(a_1, b_1, acc[1][1]); \
        asm volatile("" : "+v"(acc[0][0]), "+v"(acc[0][1]), "+v"(acc[1][0]), "+v"(acc[1][1])); \
        SCHED_FENCE(); \
    } while (0)

    auto read_frag = [&](Frag& f, u32 sbase, auto kk_tag) {
        constexpr int kk = decltype(kk_tag)::value;
        const u32 aa0 = (a0 ^ (u32)(kk << 6)) + sbase;
        const u32 aa1 = (a0 ^ (u32)((kk << 6) | 16)) + sbase;
        const u32 bb0 = b0 + sbase;
        const u32 bb1 = (b0 ^ 32u) + sbase;
        asm volatile("ds_read_b128 %0, %1 offset:0" : "=v"(f.a[0][0]) : "v"(aa0));
        asm volatile("ds_read_b128 %0, %1 offset:0" : "=v"(f.a[0][1]) : "v"(aa1));
        asm volatile("ds_read_b64_tr_b8 %0, %1 offset:%2" : "=v"(f.b[0][0]) : "v"(bb0), "n"(kk * 8192));
        asm volatile("ds_read_b64_tr_b8 %0, %1 offset:%2" : "=v"(f.b[0][1]) : "v"(bb0), "n"(kk * 8192 + 1024));
        asm volatile("ds_read_b64_tr_b8 %0, %1 offset:%2" : "=v"(f.b[0][2]) : "v"(bb0), "n"(kk * 8192 + 2048));
        asm volatile("ds_read_b64_tr_b8 %0, %1 offset:%2" : "=v"(f.b[0][3]) : "v"(bb0), "n"(kk * 8192 + 3072));
        asm volatile("ds_read_b64_tr_b8 %0, %1 offset:%2" : "=v"(f.b[1][0]) : "v"(bb1), "n"(kk * 8192));
        asm volatile("ds_read_b64_tr_b8 %0, %1 offset:%2" : "=v"(f.b[1][1]) : "v"(bb1), "n"(kk * 8192 + 1024));
        asm volatile("ds_read_b64_tr_b8 %0, %1 offset:%2" : "=v"(f.b[1][2]) : "v"(bb1), "n"(kk * 8192 + 2048));
        asm volatile("ds_read_b64_tr_b8 %0, %1 offset:%2" : "=v"(f.b[1][3]) : "v"(bb1), "n"(kk * 8192 + 3072));
        asm volatile("ds_read_b128 %0, %1 offset:4096" : "=v"(f.a[1][0]) : "v"(aa0));
        asm volatile("ds_read_b128 %0, %1 offset:4096" : "=v"(f.a[1][1]) : "v"(aa1));
    };
    using K0 = std::integral_constant<int, 0>;
    using K1 = std::integral_constant<int, 1>;

    v16f acc[2][2];
#pragma unroll
    for (int r = 0; r < 16; ++r) {
        acc[0][0][r] = 0.f; acc[0][1][r] = 0.f; acc[1][0][r] = 0.f; acc[1][1][r] = 0.f;
    }
    __builtin_amdgcn_s_barrier();
    Frag f0, f1;
    u32 sb = 0;
    const u32 fa0 = ctl + (u32)wm * 4;
    u32 fa = fa0;
    wait_word(fa, 1);
    read_frag(f0, sb, K0{});

#define TAILV 1
    for (int t = 0; t < NT - TAILV; ++t) {
        const bool wrap = (sb == (u32)((NSTAGE - 1) * STAGE));
        const u32 sbn = wrap ? 0u : sb + (u32)STAGE;
        const u32 fan = wrap ? fa0 : fa + 16u;
        read_frag(f1, sb, K1{});
        int fv;
        asm volatile("ds_read_b32 %0, %1" : "=v"(fv) : "v"(fan) : "memory");
        asm volatile("s_waitcnt lgkmcnt(13)" : FRAG_OPS(f0));
        MMA4(f0);
        asm volatile("s_waitcnt lgkmcnt(0)" : FRAG_OPS(f1), "+v"(fv));
        asm volatile("ds_write_b32 %0, %1" :: "v"(free_wr), "v"(t + 1) : "memory");
        const bool ready = (t + 1 == NT) || (__builtin_amdgcn_readfirstlane(fv) >= t + 2);
        if (!ready) {
            MMA4(f1);
            wait_word(fan, t + 2);
        }
        read_frag(f0, sbn, K0{});
        if (ready) MMA4(f1);
        sb = sbn;
        fa = fan;
    }
    float* obase = p.out + (size_t)(src * MS + rb * 256 + wm * 64 + 4 * h) * NS + cb * BN + wn * 64 + (lane & 31);
    int badv;
    read_frag(f1, sb, K1{});
    asm volatile("ds_read_b32 %0, %1" : "=v"(badv) : "v"(ctl + 128) : "memory");
    asm volatile("s_waitcnt lgkmcnt(0)" : FRAG_OPS(f0), FRAG_OPS(f1), "+v"(badv));
    const bool bad = badv != 0;
#define MMB(i, j) do { \
        SCHED_FENCE(); \
        acc[i][j] = MFMA(cat8(f0.a[i][0], f0.a[i][1]), BFRAG(f0, j), acc[i][j]); \
        acc[i][j] = MFMA(cat8(f1.a[i][0], f1.a[i][1]), BFRAG(f1, j), acc[i][j]); \
        SCHED_FENCE(); \
    } while (0)
    auto store_block = [&](const v16f& a, int i, int j) {
#pragma unroll
        for (int r = 0; r < 16; ++r) {
            float y = a[r] * scale;
            y = y > 0.f ? y : 0.f;
            if (bad) y = __builtin_nanf("");
            st_out(&obase[(size_t)(i * 32 + (r & 3) + 8 * (r >> 2)) * NS + (j ^ jsw) * 32], y);
        }
    };
    MMB(0, 0);
    MMB(0, 1);
    store_block(acc[0][0], 0, 0);
    MMB(1, 0);
    store_block(acc[0][1], 0, 1);
    MMB(1, 1);
    store_block(acc[1][0], 1, 0);
    store_block(acc[1][1], 1, 1);
}

}

extern "C" void kernel_launch(int rank, int nranks, ncclComm_t comm,
                              hipStream_t stream,
                              void* const* d_in, const int* in_sizes, int n_in,
                              void* d_out, int out_size,
                              const int* in_dtypes, int out_dtype,
                              void* const* d_out_all,
                              void* const* d_ws_all, size_t ws_bytes) {
    if (nranks != NR || ws_bytes < WS_NEED) {
        fprintf(stderr, "unsupported configuration\n");
        return;
    }
    static bool attr_set = false;
    if (!attr_set) {
        HIP_CHECK(hipFuncSetAttribute((const void*)fused_kernel,
                                      hipFuncAttributeMaxDynamicSharedMemorySize, LDS_BYTES));
        attr_set = true;
    }
    Params p;
    p.x = (const u8*)d_in[0];
    p.w = (const u8*)d_in[1];
    p.sx = (const float*)d_in[2];
    p.sw = (const float*)d_in[3];
    p.out = (float*)d_out;
    for (int i = 0; i < NR; ++i) p.ws[i] = (u8*)d_ws_all[i];
    p.rank = rank;
    hipLaunchKernelGGL(fused_kernel, dim3(NWG), dim3(NTHREADS), LDS_BYTES, stream, p);
    HIP_CHECK(hipGetLastError());
}
```
